```python
import numpy as np

kernel_name = "gpu_ag_gemm_m4096_k4096_n16384_f32_q8"


ABSMAX_RTOL = 0.02

COMPUTE_DTYPE = "float32"
EPILOGUE = "quant_int8"
IN_QUANT = None
OUT_QUANT = "int8"
IN_DTYPES = {"x": "float32", "w_mat": "float32"}
WORLD_SIZE = 8

def setup_inputs():

    x = np.random.randn(4096, 4096).astype(np.float32)
    w_mat = (np.random.randn(4096, 16384) / np.sqrt(4096)).astype(np.float32)
    return {"x": x, "w_mat": w_mat}

def _quant_dequant(y, amax):
    scale = amax / 127.0
    q = np.clip(np.round(y / scale), -127.0, 127.0)
    return (q * scale).astype(np.float32)
def _epilogue(y):
    return _quant_dequant(y, np.abs(y).max())

def reference(x, w_mat):
    y = (x @ w_mat).astype(np.float32)
    return _epilogue(y)
```

```cpp
#include <hip/hip_runtime.h>
#include <rccl/rccl.h>
#include <cstdio>

#define HIP_CHECK(expr) do { hipError_t _e = (expr); if (_e != hipSuccess) { \
    fprintf(stderr, "HIP error %s at %s:%d\n", hipGetErrorString(_e), __FILE__, __LINE__); \
    return; } } while (0)

namespace {

typedef __attribute__((ext_vector_type(4))) unsigned u32x4;
typedef __attribute__((ext_vector_type(4))) float f32x4;
typedef __attribute__((ext_vector_type(2))) float f32x2;
typedef __attribute__((ext_vector_type(2))) __bf16 bf16x2;
typedef __attribute__((ext_vector_type(8))) __bf16 bf16x8;
typedef unsigned long long u64;

constexpr int NR = 8;
constexpr int KD = 4096;
constexpr int NL = 2048;
constexpr int MS = 512;
constexpr int BM = 256, BN = 128, BK = 64;
constexpr int NKT = KD / BK;
#define NA 3
constexpr int A_STAGE = BM * BK * 2;
constexpr int B_STAGE = BN * BK * 2;
constexpr int LDS_BYTES = NA * A_STAGE + 2 * B_STAGE;
constexpr int NWG = 256;
constexpr int NTHREADS = 768;

constexpr size_t WS_XB = 0;
constexpr size_t XB_BYTES = (size_t)MS * KD * 2;
constexpr size_t WS_XFLAG = XB_BYTES;
constexpr size_t WS_GMAX = WS_XFLAG + 8192;
constexpr size_t WS_LMAX = WS_GMAX + 4096;
constexpr size_t WS_CTR = WS_LMAX + 4096;
constexpr size_t WS_TMO = WS_CTR + 256;
constexpr size_t WS_DBG = WS_TMO + 256 + 3584;
constexpr size_t WS_END = WS_DBG + 65536;

constexpr u64 DEADLINE = 100000000ull;

struct Params {
  const float* x;
  const float* w;
  signed char* out;
  char* ws[NR];
  int rank;
};

__device__ __forceinline__ __amdgpu_buffer_rsrc_t mk_rsrc(const void* p, unsigned bytes) {
  return __builtin_amdgcn_make_buffer_rsrc((void*)p, 0, bytes, 0x00020000);
}
__device__ __forceinline__ unsigned pk_bf16(float a, float b) {
  f32x2 v = {a, b};
  return __builtin_bit_cast(unsigned, __builtin_convertvector(v, bf16x2));
}
__device__ __forceinline__ void bar() { asm volatile("s_barrier" ::: "memory"); }
template <int N> __device__ __forceinline__ void wait_vm() {
  asm volatile("s_waitcnt vmcnt(%0)" ::"i"(N) : "memory");
}
template <int N> __device__ __forceinline__ void wait_vm_lgkm0() {
  asm volatile("s_waitcnt vmcnt(%0) lgkmcnt(0)" ::"i"(N) : "memory");
}
__device__ __forceinline__ void wait_lgkm0() { asm volatile("s_waitcnt lgkmcnt(0)" ::: "memory"); }

__device__ __forceinline__ bool timed_out(unsigned* tmo, unsigned call, u64 t0, unsigned& spins) {
  if ((++spins & 31u) != 0u) return false;
  if (__hip_atomic_load(tmo, __ATOMIC_RELAXED, __HIP_MEMORY_SCOPE_AGENT) == call ||
      __builtin_amdgcn_s_memrealtime() - t0 > DEADLINE) {
    __hip_atomic_store(tmo, call, __ATOMIC_RELAXED, __HIP_MEMORY_SCOPE_AGENT);
    return true;
  }
  return false;
}

#define STAMP(i) do {} while (0)

__global__ void __launch_bounds__(NTHREADS) ag_gemm_q8(Params p) {
  __shared__ __attribute__((aligned(1024))) char smem[LDS_BYTES];
  const int tid = threadIdx.x;
  const int lane = tid & 63;
  const int wv = __builtin_amdgcn_readfirstlane(tid >> 6);
  const int b = blockIdx.x;
  const int grp = b & 7, gi = b >> 3;
  const int rank = p.rank;
  const int src = (rank + grp) & 7;
  const int mt = gi >> 4, nt = gi & 15;
  char* const myws = p.ws[rank];
  unsigned* const tmo = (unsigned*)(myws + WS_TMO);
  char* const sB = smem + NA * A_STAGE;

  STAMP(0);
  const unsigned call =
      __hip_atomic_load((unsigned*)(myws + WS_CTR), __ATOMIC_RELAXED, __HIP_MEMORY_SCOPE_AGENT) + 1u;

  if (wv >= 8) {
    const int pw = wv - 8;
    const int cg = lane & 31, oct = pw * 2 + (lane >> 5);
    const __amdgpu_buffer_rsrc_t rw = mk_rsrc(p.w, (unsigned)((size_t)KD * NL * 4));
    const int voff = oct * 8 * (NL * 4) + nt * (BN * 4) + cg * 16;
    const int wofs = (cg >> 4) * 8192 + (oct >> 2) * 1024 + ((oct & 3) * 16 + (cg & 15)) * 16;
    f32x4 R0[8], R1[8];
#define LOADB(R, kt) do { const int _s = (kt) * (BK * NL * 4); _Pragma("unroll") for (int r = 0; r < 8; ++r) \
      R[r] = __builtin_bit_cast(f32x4, __builtin_amdgcn_raw_buffer_load_b128(rw, voff, _s + r * (NL * 4), 0)); } while (0)
#define WRITEB(R, st) do { _Pragma("unroll") for (int t = 0; t < 4; ++t) { u32x4 v; \
      v[0] = pk_bf16(R[0][t], R[1][t]); v[1] = pk_bf16(R[2][t], R[3][t]); \
      v[2] = pk_bf16(R[4][t], R[5][t]); v[3] = pk_bf16(R[6][t], R[7][t]); \
      *(u32x4*)(sB + (st) * B_STAGE + wofs + t * 2048) = v; } } while (0)
    LOADB(R0, 0);
    LOADB(R1, 1);
    bar();
    bar();
    WRITEB(R0, 0);
    LOADB(R0, 2);
    for (int t = 0; t < NKT; t += 2) {
      wait_lgkm0(); bar();
      WRITEB(R1, 1);
      if (t + 3 < NKT) LOADB(R1, t + 3);
      wait_lgkm0(); bar();
      if (t + 2 < NKT) { WRITEB(R0, 0); }
      if (t + 4 < NKT) LOADB(R0, t + 4);
    }
    return;
  }

  if (tid == 0) __builtin_amdgcn_fence(__ATOMIC_ACQUIRE, "");
  {
    const int pm = b >> 7, pk = (b & 127) >> 1, mb = (b & 1) * 8 + wv;
    const float* s = p.x + (size_t)(pm * BM + mb * 16 + (lane & 15)) * KD + pk * BK + (lane >> 4) * 8;
    const f32x4 a0 = *(const f32x4*)(s), a1 = *(const f32x4*)(s + 4);
    const f32x4 c0 = *(const f32x4*)(s + 32), c1 = *(const f32x4*)(s + 36);
    u32x4 v0, v1;
    v0[0] = pk_bf16(a0[0], a0[1]); v0[1] = pk_bf16(a0[2], a0[3]);
    v0[2] = pk_bf16(a1[0], a1[1]); v0[3] = pk_bf16(a1[2], a1[3]);
    v1[0] = pk_bf16(c0[0], c0[1]); v1[1] = pk_bf16(c0[2], c0[3]);
    v1[2] = pk_bf16(c1[0], c1[1]); v1[3] = pk_bf16(c1[2], c1[3]);
    const __amdgpu_buffer_rsrc_t rx = mk_rsrc(myws + WS_XB, (unsigned)XB_BYTES);
    const int o = (pm * NKT + pk) * A_STAGE + mb * 2048 + lane * 16;
    __builtin_amdgcn_raw_buffer_store_b128(v0, rx, o, 0, 17);
    __builtin_amdgcn_raw_buffer_store_b128(v1, rx, o + 1024, 0, 17);
    wait_vm<0>();
  }
  bar();
  if (wv == 0) {
    if (lane < NR)
      __hip_atomic_store((unsigned*)(p.ws[lane] + WS_XFLAG) + rank * NWG + b, call,
                         __ATOMIC_RELAXED, __HIP_MEMORY_SCOPE_SYSTEM);
    STAMP(1);
    u64* f = (u64*)(myws + WS_XFLAG) + (src * NWG + mt * 128) / 2 + lane;
    const u64 t0 = __builtin_amdgcn_s_memrealtime();
    unsigned spins = 0;
    for (;;) {
      const u64 v = __hip_atomic_load(f, __ATOMIC_RELAXED, __HIP_MEMORY_SCOPE_SYSTEM);
      const bool ok = (unsigned)v >= call && (unsigned)(v >> 32) >= call;
      if (__all(ok)) break;
      if (timed_out(tmo, call, t0, spins)) break;
      __builtin_amdgcn_s_sleep(1);
    }
    wait_vm<0>();
    STAMP(2);
  }
  bar();

  const int wm = wv >> 1, wn = wv & 1;
  const __amdgpu_buffer_rsrc_t ra =
      mk_rsrc(p.ws[src] + WS_XB + (size_t)mt * NKT * A_STAGE, (unsigned)(NKT * A_STAGE));
  const int avoff = wv * 4096 + lane * 16;
#define STAGEA1(kt, st, o) __builtin_amdgcn_raw_ptr_buffer_load_lds(ra, \
    (__attribute__((address_space(3))) void*)(smem + (st) * A_STAGE + wv * 4096), 16, avoff, (kt) * A_STAGE, o, 0)
#define STAGEA(kt, st) do { STAGEA1(kt, st, 0); STAGEA1(kt, st, 1024); STAGEA1(kt, st, 2048); STAGEA1(kt, st, 3072); } while (0)
#pragma unroll
  for (int s = 0; s < NA - 1; ++s) STAGEA(s, s);

  f32x4 acc[4][4];
#pragma unroll
  for (int i = 0; i < 4; ++i)
#pragma unroll
    for (int j = 0; j < 4; ++j) acc[i][j] = (f32x4){0.f, 0.f, 0.f, 0.f};

  const int aro = wm * 8192 + lane * 16;
  const int bro = NA * A_STAGE + wn * 8192 + lane * 16;
  int sa = 0;
#define COMPUTE(t) do { \
    const char* pa = smem + aro + sa * A_STAGE; \
    const char* pb = smem + bro + ((t) & 1) * B_STAGE; \
    _Pragma("unroll") for (int kh = 0; kh < 2; ++kh) { \
      bf16x8 xa[4], wb[4]; \
      _Pragma("unroll") for (int i = 0; i < 4; ++i) wb[i] = *(const bf16x8*)(pb + i * 2048 + kh * 1024); \
      _Pragma("unroll") for (int i = 0; i < 4; ++i) xa[i] = *(const bf16x8*)(pa + i * 2048 + kh * 1024); \
      _Pragma("unroll") for (int i = 0; i < 4; ++i) \
        _Pragma("unroll") for (int j = 0; j < 4; ++j) \
          acc[i][j] = __builtin_amdgcn_mfma_f32_16x16x32_bf16(wb[j], xa[i], acc[i][j], 0, 0, 0); \
    } } while (0)

  for (int t = 0; t < NKT - (NA - 1); ++t) {
    wait_vm_lgkm0<4 * (NA - 2)>();
    bar();
    const int sn = sa == 0 ? NA - 1 : sa - 1;
    STAGEA(t + NA - 1, sn);
    COMPUTE(t);
    sa = sa == NA - 1 ? 0 : sa + 1;
  }
#pragma unroll
  for (int r = NA - 2; r >= 0; --r) {
    if (r == 2) wait_vm_lgkm0<8>(); else if (r == 1) wait_vm_lgkm0<4>(); else wait_vm_lgkm0<0>();
    bar();
    COMPUTE(NKT - 1 - r);
    sa = sa == NA - 1 ? 0 : sa + 1;
  }
  STAMP(3);

  float m = 0.f;
#pragma unroll
  for (int i = 0; i < 4; ++i)
#pragma unroll
    for (int j = 0; j < 4; ++j)
#pragma unroll
      for (int r = 0; r < 4; ++r) m = fmaxf(m, fabsf(acc[i][j][r]));
  unsigned mu = __builtin_bit_cast(unsigned, m);
#pragma unroll
  for (int o = 32; o >= 1; o >>= 1) mu = max(mu, (unsigned)__shfl_xor((int)mu, o));
  unsigned* red = (unsigned*)(smem + NA * A_STAGE);
  if (lane == 0) red[wv] = mu;
  wait_lgkm0();
  bar();
  if (wv == 0) {
    mu = red[lane & 7];
#pragma unroll
    for (int o = 4; o >= 1; o >>= 1) mu = max(mu, (unsigned)__shfl_xor((int)mu, o));
    const u64 t0 = __builtin_amdgcn_s_memrealtime();
    unsigned spins = 0;
    u64* lm = (u64*)(myws + WS_LMAX) + grp * 32;
    if (gi != 0) {
      if (lane == 0)
        __hip_atomic_store(lm + gi, ((u64)call << 32) | mu, __ATOMIC_RELAXED, __HIP_MEMORY_SCOPE_AGENT);
    } else {
      for (;;) {
        u64 v = ((u64)call << 32);
        if (lane >= 1 && lane < 32) v = __hip_atomic_load(lm + lane, __ATOMIC_RELAXED, __HIP_MEMORY_SCOPE_AGENT);
        if (__all((unsigned)(v >> 32) == call)) { mu = max(mu, (unsigned)v); break; }
        if (timed_out(tmo, call, t0, spins)) break;
      }
#pragma unroll
      for (int o = 16; o >= 1; o >>= 1) mu = max(mu, (unsigned)__shfl_xor((int)mu, o));
      if (lane < NR)
        __hip_atomic_store((u64*)(p.ws[lane] + WS_GMAX) + rank * 8 + grp, ((u64)call << 32) | mu,
                           __ATOMIC_RELAXED, __HIP_MEMORY_SCOPE_SYSTEM);
    }
    STAMP(4);
    u64* gm = (u64*)(myws + WS_GMAX) + lane;
    bool bad = false;
    for (;;) {
      const u64 v = __hip_atomic_load(gm, __ATOMIC_RELAXED, __HIP_MEMORY_SCOPE_SYSTEM);
      if (__all((unsigned)(v >> 32) == call)) { mu = (unsigned)v; break; }
      if (timed_out(tmo, call, t0, spins)) { bad = true; break; }
      __builtin_amdgcn_s_sleep(1);
    }
#pragma unroll
    for (int o = 32; o >= 1; o >>= 1) mu = max(mu, (unsigned)__shfl_xor((int)mu, o));
    if (__hip_atomic_load(tmo, __ATOMIC_RELAXED, __HIP_MEMORY_SCOPE_AGENT) == call) bad = true;
    if (lane == 0) { red[16] = mu; red[17] = bad ? 1u : 0u; }
    if (b == 0 && lane == 0)
      __hip_atomic_store((unsigned*)(myws + WS_CTR), call, __ATOMIC_RELAXED, __HIP_MEMORY_SCOPE_AGENT);
    wait_lgkm0();
    STAMP(5);
  }
  bar();
  const float amax = __builtin_bit_cast(float, red[16]);
  const bool bad = red[17] != 0;
  const float inv = amax > 0.f ? 127.0f / amax : 0.f;

  signed char* o = p.out + (size_t)(src * MS + mt * BM + wm * 64 + (lane & 15)) * NL + nt * BN + wn * 64 + (lane >> 4) * 16;
#pragma unroll
  for (int i = 0; i < 4; ++i) {
    u32x4 v;
#pragma unroll
    for (int r = 0; r < 4; ++r) {
      unsigned d = 0;
#pragma unroll
      for (int j = 0; j < 4; ++j) {
        float q = __builtin_rintf(acc[i][j][r] * inv);
        q = fminf(fmaxf(q, -127.f), 127.f);
        d |= ((unsigned)(int)q & 255u) << (8 * j);
      }
      v[r] = bad ? 0x80808080u : d;
    }
    *(u32x4*)(o + (size_t)i * 16 * NL) = v;
  }
  STAMP(6);
}

}

extern "C" void kernel_launch(int rank, int nranks, ncclComm_t comm, hipStream_t stream,
                              void* const* d_in, const int* in_sizes, int n_in, void* d_out,
                              int out_size, const int* in_dtypes, int out_dtype,
                              void* const* d_out_all, void* const* d_ws_all, size_t ws_bytes) {
  (void)comm; (void)in_dtypes; (void)out_dtype; (void)d_out_all;
  if (nranks != NR || (unsigned)rank >= (unsigned)NR || n_in < 2 || ws_bytes < WS_END ||
      in_sizes[0] != MS * KD || in_sizes[1] != KD * NL || out_size != NR * MS * NL) {
    fprintf(stderr, "unexpected problem shape\n");
    return;
  }
  Params p;
  p.x = (const float*)d_in[0];
  p.w = (const float*)d_in[1];
  p.out = (signed char*)d_out;
  for (int i = 0; i < NR; ++i) p.ws[i] = (char*)d_ws_all[i];
  p.rank = rank;
  hipLaunchKernelGGL(ag_gemm_q8, dim3(NWG), dim3(NTHREADS), 0, stream, p);
  HIP_CHECK(hipGetLastError());
}
```

```cpp
#include <hip/hip_runtime.h>
#include <rccl/rccl.h>
#include <cstdio>

#define HIP_CHECK(expr) do { hipError_t _e = (expr); if (_e != hipSuccess) { \
    fprintf(stderr, "HIP error %s at %s:%d\n", hipGetErrorString(_e), __FILE__, __LINE__); \
    return; } } while (0)

namespace {

typedef __attribute__((ext_vector_type(4))) unsigned u32x4;
typedef __attribute__((ext_vector_type(2))) unsigned u32x2;
typedef __attribute__((ext_vector_type(4))) float f32x4;
typedef __attribute__((ext_vector_type(2))) float f32x2;
typedef __attribute__((ext_vector_type(2))) __bf16 bf16x2;
typedef __attribute__((ext_vector_type(8))) __bf16 bf16x8;
typedef unsigned long long u64;

constexpr int NR = 8;
constexpr int KD = 4096;
constexpr int NL = 2048;
constexpr int MS = 512;
constexpr int BM = 256, BN = 128, BK = 64;
constexpr int NKT = KD / BK;
constexpr int NTILE = 2 * NKT;
#define NA 3
#define NB 3
#define UPT 1
constexpr int NUNIT = NTILE * UPT;
#define TAU (884 / UPT)
#define BACKLOG (2 * UPT)
#define LEAD 140
#define SMOOTH 3
#define AHEAD 7
#define HAUX 3
constexpr int A_STAGE = BM * BK * 2;
constexpr int B_STAGE = BN * BK * 2;
constexpr int LDS_CTL = NA * A_STAGE + NB * B_STAGE;
constexpr int LDS_BYTES = LDS_CTL + 512;
#define TR(k, i) do {} while (0)
constexpr int NWG = 256;
constexpr int NTHREADS = 960;

constexpr size_t WS_XB = 0;
constexpr size_t XB_BYTES = (size_t)MS * KD * 2;
constexpr size_t WS_XFLAG = XB_BYTES;
constexpr size_t WS_GMAX = WS_XFLAG + 8192;
constexpr size_t WS_LMAX = WS_GMAX + 4096;
constexpr size_t WS_HINT = WS_LMAX + 4096;
constexpr size_t WS_CTR = WS_HINT + 16384;
constexpr size_t WS_TMO = WS_CTR + 256;
constexpr size_t WS_DBG = WS_TMO + 256 + 3584;
constexpr size_t WS_END = WS_DBG + 65536 + 256 * 2048;

constexpr u64 DEADLINE = 100000000ull;
constexpr u64 HINT_WAIT = 10000ull;

struct Params {
  const float* x;
  const float* w;
  signed char* out;
  char* ws[NR];
  int rank;
};

typedef __attribute__((address_space(3))) unsigned lds_u32;

__device__ __forceinline__ __amdgpu_buffer_rsrc_t mk_rsrc(const void* p, unsigned bytes) {
  return __builtin_amdgcn_make_buffer_rsrc((void*)p, 0, bytes, 0x00020000);
}
__device__ __forceinline__ unsigned pk_bf16(float a, float b) {
  f32x2 v = {a, b};
  return __builtin_bit_cast(unsigned, __builtin_convertvector(v, bf16x2));
}
__device__ __forceinline__ void bar() { asm volatile("s_barrier" ::: "memory"); }
__device__ __forceinline__ void fence_c() { asm volatile("" ::: "memory"); }
template <int N> __device__ __forceinline__ void wait_vm() {
  asm volatile("s_waitcnt vmcnt(%0)" ::"i"(N) : "memory");
}
__device__ __forceinline__ u64 now() { return __builtin_amdgcn_s_memrealtime(); }
__device__ __forceinline__ void give_up(unsigned* tmo, unsigned call) {
  __hip_atomic_store(tmo, call, __ATOMIC_RELAXED, __HIP_MEMORY_SCOPE_AGENT);
}
__device__ __forceinline__ bool timed_out(unsigned* tmo, unsigned call, u64 t0, unsigned& spins) {
  if ((++spins & 31u) != 0u) return false;
  if (__hip_atomic_load(tmo, __ATOMIC_RELAXED, __HIP_MEMORY_SCOPE_AGENT) == call || now() - t0 > DEADLINE) {
    give_up(tmo, call);
    return true;
  }
  return false;
}
__device__ __forceinline__ void lds_wait(volatile lds_u32* w, unsigned need, unsigned* tmo, unsigned call) {
  if (__all((int)(*w - need) >= 0)) return;
  const u64 t0 = now();
  unsigned spins = 0;
  while (!__all((int)(*w - need) >= 0)) {
    __builtin_amdgcn_s_sleep(1);
    if ((++spins & 1023u) == 0u && now() - t0 > 2 * DEADLINE) { give_up(tmo, call); break; }
  }
}
__device__ __forceinline__ void wait_src(char* myws, int src, int j, unsigned* tmo, unsigned call) {
  u64* f = (u64*)(myws + WS_XFLAG) + src * NTILE + j;
  const u64 t0 = now();
  unsigned spins = 0;
  for (;;) {
    const u64 v = __hip_atomic_load(f, __ATOMIC_RELAXED, __HIP_MEMORY_SCOPE_SYSTEM);
    if (__all((unsigned)v >= call && (unsigned)(v >> 32) >= call)) break;
    if (timed_out(tmo, call, t0, spins)) break;
    __builtin_amdgcn_s_sleep(1);
  }
}

#define STAMP(i) do {} while (0)

__global__ void __launch_bounds__(NTHREADS) ag_gemm_q8(Params p) {
  __shared__ __attribute__((aligned(1024))) char smem[LDS_BYTES];
  const int tid = threadIdx.x;
  const int lane = tid & 63;
  const int wv = __builtin_amdgcn_readfirstlane(tid >> 6);
  const int b = blockIdx.x;
  const int rank = p.rank;
  const int grp = b & 7, gi = b >> 3;
  const int src = (rank + grp) & 7;
  const int mt = gi >> 4, nt = gi & 15;
  char* const myws = p.ws[rank];
  unsigned* const tmo = (unsigned*)(myws + WS_TMO);
  unsigned* const hint = (unsigned*)(myws + WS_HINT) + grp * 512;
  char* const sB = smem + NA * A_STAGE;
  volatile lds_u32* const ctl = (volatile lds_u32*)(smem + LDS_CTL);

  if (wv == 0) STAMP(0);
  const unsigned ctr = __hip_atomic_load((unsigned*)(myws + WS_CTR), __ATOMIC_RELAXED, __HIP_MEMORY_SCOPE_AGENT);
#define GET_CALL() const unsigned call = __builtin_amdgcn_readfirstlane(ctr) + 1u

  if (wv >= 8 && wv < 12) {
    const int pw = wv - 8;
    const int cg = lane & 31, oct = pw * 2 + (lane >> 5);
    const __amdgpu_buffer_rsrc_t rw = mk_rsrc(p.w, (unsigned)((size_t)KD * NL * 4));
    const int voff = oct * 8 * (NL * 4) + nt * (BN * 4) + cg * 16;
    const int wofs = (cg >> 4) * 8192 + (oct >> 2) * 1024 + ((oct & 3) * 16 + (cg & 15)) * 16;
    f32x4 R0[8], R1[8];
#define LOADB(R, kt) do { const int _s = (kt) * (BK * NL * 4); _Pragma("unroll") for (int r = 0; r < 8; ++r) \
      R[r] = __builtin_bit_cast(f32x4, __builtin_amdgcn_raw_buffer_load_b128(rw, voff, _s + r * (NL * 4), 0)); } while (0)
#define WRITEB(R, st) do { _Pragma("unroll") for (int t = 0; t < 4; ++t) { u32x4 v; \
      v[0] = pk_bf16(R[0][t], R[1][t]); v[1] = pk_bf16(R[2][t], R[3][t]); \
      v[2] = pk_bf16(R[4][t], R[5][t]); v[3] = pk_bf16(R[6][t], R[7][t]); \
      *(u32x4*)(sB + (st) * B_STAGE + wofs + t * 2048) = v; } } while (0)
    LOADB(R0, 0);
    LOADB(R1, 1);
    bar();
    GET_CALL();
    int sb = 0;
#define PUTB(R, t) do { lds_wait(ctl + (lane & 7), (t) + 1 - NB, tmo, call); WRITEB(R, sb); \
      fence_c(); ctl[8 + pw] = (t) + 1; if (pw == 0) TR(5, t); fence_c(); sb = sb == NB - 1 ? 0 : sb + 1; } while (0)
    for (int t = 0; t < NKT - 2; t += 2) {
      PUTB(R0, t);
      LOADB(R0, t + 2);
      PUTB(R1, t + 1);
      LOADB(R1, t + 3);
    }
    PUTB(R0, NKT - 2);
    PUTB(R1, NKT - 1);
    return;
  }

  if (wv == 12) {
    if (lane < 32) ctl[lane] = 0;
    asm volatile("s_waitcnt lgkmcnt(0)" ::: "memory");
    bar();
    GET_CALL();
    const __amdgpu_buffer_rsrc_t ra = mk_rsrc(p.ws[src] + WS_XB, (unsigned)XB_BYTES);
    const int avoff = lane * 16;
    volatile lds_u32* const look = ctl + (lane < 8 ? lane : 13);
    const unsigned slack = lane < 8 ? NA : 0;
    int I = 0, S = 0, si = 0;
    u64 t0 = now();
    unsigned spins = 0;
    while (S < NKT) {
      if (I < NKT && I - S < 2 && __all((int)(*look + slack - (unsigned)(I + 1)) >= 0)) {
        const int so = (I * 2 + mt) * A_STAGE;
#pragma unroll
        for (int c = 0; c < 8; ++c) {
          __attribute__((address_space(3))) void* d =
              (__attribute__((address_space(3))) void*)(smem + si * A_STAGE + c * 4096);
          __builtin_amdgcn_raw_ptr_buffer_load_lds(ra, d, 16, avoff, so + c * 4096, 0, 0);
          __builtin_amdgcn_raw_ptr_buffer_load_lds(ra, d, 16, avoff, so + c * 4096, 1024, 0);
          __builtin_amdgcn_raw_ptr_buffer_load_lds(ra, d, 16, avoff, so + c * 4096, 2048, 0);
          __builtin_amdgcn_raw_ptr_buffer_load_lds(ra, d, 16, avoff, so + c * 4096, 3072, 0);
        }
        TR(1, I);
        ++I;
        si = si == NA - 1 ? 0 : si + 1;
      } else if (S < I) {
        if (I - S == 2) wait_vm<32>(); else wait_vm<0>();
        TR(2, S);
        ++S;
        ctl[12] = S;
        t0 = now();
      } else {
        __builtin_amdgcn_s_sleep(1);
        if ((++spins & 1023u) == 0u && now() - t0 > 2 * DEADLINE) { give_up(tmo, call); ctl[12] = NKT; break; }
      }
    }
    return;
  }

  if (wv == 13) {
    __builtin_amdgcn_fence(__ATOMIC_ACQUIRE, "");
    wait_vm<0>();
    bar();
    GET_CALL();
    const __amdgpu_buffer_rsrc_t ra = mk_rsrc(p.ws[src] + WS_XB, (unsigned)XB_BYTES);
    const __amdgpu_buffer_rsrc_t rh = mk_rsrc(hint, NUNIT * 4);
    for (int q = gi; q < NUNIT; q += 32) {
      wait_src(myws, src, q / UPT, tmo, call);
      const u64 t0 = now();
      unsigned spins = 0;
      for (;;) {
        const unsigned ph = ctl[14];
        if (ph != 0u && (int)((unsigned)now() - ph - (unsigned)(q * TAU / 16)) >= 0 &&
            q / (2 * UPT) - (int)ctl[0] <= AHEAD) break;
        __builtin_amdgcn_s_sleep(1);
        if ((++spins & 255u) == 0u && now() - t0 > 2 * HINT_WAIT) break;
      }
      TR(6, q >> 5);
#pragma unroll
      for (int u = 0; u < 4 / UPT; ++u)
        __builtin_amdgcn_raw_ptr_buffer_load_lds(ra, (__attribute__((address_space(3))) void*)(smem + LDS_CTL + 256),
                                                 4, lane * 128, q * (A_STAGE / UPT) + u * 8192, 0, 0);
      wait_vm<0>();
      if (lane == 0) __builtin_amdgcn_raw_buffer_store_b32(call, rh, q * 4, 0, 0);
      TR(6, 16 + (q >> 5));
    }
    return;
  }

  if (wv == 14) {
    bar();
    GET_CALL();
    const __amdgpu_buffer_rsrc_t rh = mk_rsrc(hint, NUNIT * 4);
    wait_src(myws, src, lane & 1, tmo, call);
    const unsigned base = (unsigned)now();
    int ph = -BACKLOG * TAU;
    ctl[14] = (base + ph / 16) | 1u;
    int R = 0, strikes = 0;
    u64 t0 = now();
    unsigned spins = 0;
    while (R < NKT) {
      fence_c();
      const unsigned ts = (unsigned)now();
      const u32x2 v = __builtin_amdgcn_raw_buffer_load_b64(rh, lane * 8, 0, HAUX);
      const u64 m0 = __ballot(v[0] == call), m1 = __ballot(v[1] == call);
      const u64 all = ~(mt ? m1 : m0);
      const int n = __builtin_popcountll(m0) + __builtin_popcountll(m1);
      const int r = all ? __builtin_ctzll(all) : NKT;
      if (n > 0) {
        const int el = min((int)(ts - base), 1 << 24);
        ph += ((el - LEAD) * 16 - n * TAU - ph) >> SMOOTH;
        ctl[14] = (base + ph / 16) | 1u;
      }
      if (r > R) {
        R = r; ctl[13] = R;
        t0 = now(); strikes = 0;
      } else if ((++spins & 63u) == 0u && now() - t0 > HINT_WAIT) {
        wait_src(myws, src, R * 2 + mt, tmo, call);
        if (++strikes == 2) break;
        t0 = now();
      }
    }
    for (; R < NKT; ++R) {
      wait_src(myws, src, R * 2 + mt, tmo, call);
      ctl[13] = R + 1;
    }
    return;
  }

  char* peer = p.ws[lane & 7];
  {
    const int pj = b >> 1, pm = pj & 1, pk = pj >> 1, mb = (b & 1) * 8 + wv;
    const float* s = p.x + (size_t)(pm * BM + mb * 16 + (lane & 15)) * KD + pk * BK + (lane >> 4) * 8;
    const f32x4 a0 = *(const f32x4*)(s), a1 = *(const f32x4*)(s + 4);
    const f32x4 c0 = *(const f32x4*)(s + 32), c1 = *(const f32x4*)(s + 36);
    u32x4 v0, v1;
    v0[0] = pk_bf16(a0[0], a0[1]); v0[1] = pk_bf16(a0[2], a0[3]);
    v0[2] = pk_bf16(a1[0], a1[1]); v0[3] = pk_bf16(a1[2], a1[3]);
    v1[0] = pk_bf16(c0[0], c0[1]); v1[1] = pk_bf16(c0[2], c0[3]);
    v1[2] = pk_bf16(c1[0], c1[1]); v1[3] = pk_bf16(c1[2], c1[3]);
    const __amdgpu_buffer_rsrc_t rx = mk_rsrc(myws + WS_XB, (unsigned)XB_BYTES);
    const int o = pj * A_STAGE + mb * 2048 + lane * 16;
    __builtin_amdgcn_raw_buffer_store_b128(v0, rx, o, 0, 17);
    __builtin_amdgcn_raw_buffer_store_b128(v1, rx, o + 1024, 0, 17);
    asm volatile("" : "+v"(peer));
    wait_vm<0>();
  }
  bar();
  GET_CALL();
  if (wv == 0) {
    if (lane < NR)
      __hip_atomic_store((unsigned*)(peer + WS_XFLAG) + rank * NWG + b, call,
                         __ATOMIC_RELAXED, __HIP_MEMORY_SCOPE_SYSTEM);
    STAMP(1);
  }

  const int wm = wv >> 1, wn = wv & 1;
  f32x4 acc[4][4];
#pragma unroll
  for (int i = 0; i < 4; ++i)
#pragma unroll
    for (int j = 0; j < 4; ++j) acc[i][j] = (f32x4){0.f, 0.f, 0.f, 0.f};

  const char* const pa0 = smem + wm * 8192 + lane * 16;
  const char* const pb0 = smem + NA * A_STAGE + wn * 8192 + lane * 16;
  volatile lds_u32* const rdy = ctl + 8 + (lane < 4 ? lane : 4);
  int sa = 0, sb = 0;
  for (int t = 0; t < NKT; ++t) {
    lds_wait(rdy, t + 1, tmo, call);
    if (wv == 0) TR(3, t);
    fence_c();
    const char* pa = pa0 + sa * A_STAGE;
    const char* pb = pb0 + sb * B_STAGE;
    bf16x8 w0[4], w1[4], x0[4], x1[4];
#pragma unroll
    for (int i = 0; i < 4; ++i) w0[i] = *(const bf16x8*)(pb + i * 2048);
#pragma unroll
    for (int i = 0; i < 4; ++i) x0[i] = *(const bf16x8*)(pa + i * 2048);
#pragma unroll
    for (int i = 0; i < 4; ++i) w1[i] = *(const bf16x8*)(pb + i * 2048 + 1024);
    __builtin_amdgcn_sched_barrier(0);
#pragma unroll
    for (int i = 0; i < 4; ++i) {
#pragma unroll
      for (int j = 0; j < 4; ++j)
        acc[i][j] = __builtin_amdgcn_mfma_f32_16x16x32_bf16(w0[j], x0[i], acc[i][j], 0, 0, 0);
      __builtin_amdgcn_sched_barrier(0);
      x1[i] = *(const bf16x8*)(pa + i * 2048 + 1024);
      __builtin_amdgcn_sched_barrier(0);
    }
    fence_c();
    ctl[wv] = t + 1;
    fence_c();
    __builtin_amdgcn_sched_barrier(0);
#pragma unroll
    for (int i = 0; i < 4; ++i)
#pragma unroll
      for (int j = 0; j < 4; ++j)
        acc[i][j] = __builtin_amdgcn_mfma_f32_16x16x32_bf16(w1[j], x1[i], acc[i][j], 0, 0, 0);
    __builtin_amdgcn_sched_barrier(0);
    if (wv == 0) TR(4, t);
    sa = sa == NA - 1 ? 0 : sa + 1;
    sb = sb == NB - 1 ? 0 : sb + 1;
  }
  if (wv == 0) STAMP(3);

  float m = 0.f;
#pragma unroll
  for (int i = 0; i < 4; ++i)
#pragma unroll
    for (int j = 0; j < 4; ++j)
#pragma unroll
      for (int r = 0; r < 4; ++r) m = fmaxf(m, fabsf(acc[i][j][r]));
  unsigned mu = __builtin_bit_cast(unsigned, m);
#pragma unroll
  for (int o = 32; o >= 1; o >>= 1) mu = max(mu, (unsigned)__shfl_xor((int)mu, o));
  if (lane == 0) ctl[16 + wv] = mu | 0x80000000u;
  if (wv == 0) {
    lds_wait(ctl + 16 + (lane & 7), 0x80000000u, tmo, call);
    mu = ctl[16 + (lane & 7)] & 0x7fffffffu;
#pragma unroll
    for (int o = 4; o >= 1; o >>= 1) mu = max(mu, (unsigned)__shfl_xor((int)mu, o));
    const u64 t0 = now();
    unsigned spins = 0;
    u64* lm = (u64*)(myws + WS_LMAX) + grp * 32;
    if (gi != 0) {
      if (lane == 0)
        __hip_atomic_store(lm + gi, ((u64)call << 32) | mu, __ATOMIC_RELAXED, __HIP_MEMORY_SCOPE_AGENT);
    } else {
      for (;;) {
        u64 v = ((u64)call << 32);
        if (lane >= 1 && lane < 32) v = __hip_atomic_load(lm + lane, __ATOMIC_RELAXED, __HIP_MEMORY_SCOPE_AGENT);
        if (__all((unsigned)(v >> 32) == call)) { mu = max(mu, (unsigned)v); break; }
        if (timed_out(tmo, call, t0, spins)) break;
      }
#pragma unroll
      for (int o = 16; o >= 1; o >>= 1) mu = max(mu, (unsigned)__shfl_xor((int)mu, o));
      if (lane < NR)
        __hip_atomic_store((u64*)(peer + WS_GMAX) + rank * 8 + grp, ((u64)call << 32) | mu,
                           __ATOMIC_RELAXED, __HIP_MEMORY_SCOPE_SYSTEM);
    }
    STAMP(4);
    u64* gm = (u64*)(myws + WS_GMAX) + lane;
    bool bad = false;
    for (;;) {
      const u64 v = __hip_atomic_load(gm, __ATOMIC_RELAXED, __HIP_MEMORY_SCOPE_SYSTEM);
      if (__all((unsigned)(v >> 32) == call)) { mu = (unsigned)v; break; }
      if (timed_out(tmo, call, t0, spins)) { bad = true; break; }
      __builtin_amdgcn_s_sleep(1);
    }
#pragma unroll
    for (int o = 32; o >= 1; o >>= 1) mu = max(mu, (unsigned)__shfl_xor((int)mu, o));
    if (__hip_atomic_load(tmo, __ATOMIC_RELAXED, __HIP_MEMORY_SCOPE_AGENT) == call) bad = true;
    if (lane == 0) { ctl[24] = mu; ctl[25] = bad ? 2u : 1u; }
    if (b == 0 && lane == 0)
      __hip_atomic_store((unsigned*)(myws + WS_CTR), call, __ATOMIC_RELAXED, __HIP_MEMORY_SCOPE_AGENT);
    STAMP(5);
  }
  lds_wait(ctl + 25, 1, tmo, call);
  const float amax = __builtin_bit_cast(float, (unsigned)ctl[24]);
  const bool bad = ctl[25] != 1u;
  const float inv = amax > 0.f ? 127.0f / amax : 0.f;

  signed char* o = p.out + (size_t)(src * MS + mt * BM + wm * 64 + (lane & 15)) * NL + nt * BN + wn * 64 + (lane >> 4) * 16;
#pragma unroll
  for (int i = 0; i < 4; ++i) {
    u32x4 v;
#pragma unroll
    for (int r = 0; r < 4; ++r) {
      unsigned d = 0;
#pragma unroll
      for (int j = 0; j < 4; ++j) {
        float q = __builtin_rintf(acc[i][j][r] * inv);
        q = fminf(fmaxf(q, -127.f), 127.f);
        d |= ((unsigned)(int)q & 255u) << (8 * j);
      }
      v[r] = bad ? 0x80808080u : d;
    }
    *(u32x4*)(o + (size_t)i * 16 * NL) = v;
  }
  if (wv == 0) STAMP(6);
}

}

extern "C" void kernel_launch(int rank, int nranks, ncclComm_t comm, hipStream_t stream,
                              void* const* d_in, const int* in_sizes, int n_in, void* d_out,
                              int out_size, const int* in_dtypes, int out_dtype,
                              void* const* d_out_all, void* const* d_ws_all, size_t ws_bytes) {
  (void)comm; (void)in_dtypes; (void)out_dtype; (void)d_out_all;
  if (nranks != NR || (unsigned)rank >= (unsigned)NR || n_in < 2 || ws_bytes < WS_END ||
      in_sizes[0] != MS * KD || in_sizes[1] != KD * NL || out_size != NR * MS * NL) {
    fprintf(stderr, "unexpected problem shape\n");
    return;
  }
  Params p;
  p.x = (const float*)d_in[0];
  p.w = (const float*)d_in[1];
  p.out = (signed char*)d_out;
  for (int i = 0; i < NR; ++i) p.ws[i] = (char*)d_ws_all[i];
  p.rank = rank;
  hipLaunchKernelGGL(ag_gemm_q8, dim3(NWG), dim3(NTHREADS), 0, stream, p);
  HIP_CHECK(hipGetLastError());
}
```

```cpp
#include <hip/hip_runtime.h>
#include <rccl/rccl.h>
#include <cstdio>

#define HIP_CHECK(expr) do { hipError_t _e = (expr); if (_e != hipSuccess) { \
    fprintf(stderr, "HIP error %s at %s:%d\n", hipGetErrorString(_e), __FILE__, __LINE__); \
    return; } } while (0)

namespace {

typedef __attribute__((ext_vector_type(4))) unsigned u32x4;
typedef __attribute__((ext_vector_type(2))) unsigned u32x2;
typedef __attribute__((ext_vector_type(4))) float f32x4;
typedef __attribute__((ext_vector_type(2))) float f32x2;
typedef __attribute__((ext_vector_type(2))) __bf16 bf16x2;
typedef __attribute__((ext_vector_type(8))) __bf16 bf16x8;
typedef unsigned long long u64;

constexpr int NR = 8;
constexpr int KD = 4096;
constexpr int NL = 2048;
constexpr int MS = 512;
constexpr int BM = 256, BN = 128, BK = 64;
constexpr int NKT = KD / BK;
constexpr int NTILE = 2 * NKT;
#define NA 3
#define NB 3
#define UPT 2
constexpr int NUNIT = NTILE * UPT;
#define TAU (884 / UPT)
#define BACKLOG 2
#define LEAD 130
#define SMOOTH 3
#define SMOOTHD SMOOTH
#define AHEAD 7
#define WPACE 100
#define WAUX 2
#define OAUX 0
#define HAUX 3
constexpr int A_STAGE = BM * BK * 2;
constexpr int B_STAGE = BN * BK * 2;
constexpr int LDS_CTL = NA * A_STAGE + NB * B_STAGE;
constexpr int LDS_BYTES = LDS_CTL + 512;
#define TR(k, i) do {} while (0)
constexpr int NWG = 256;
constexpr int NTHREADS = 1024;

constexpr size_t WS_XB = 0;
constexpr size_t XB_BYTES = (size_t)MS * KD * 2;
constexpr size_t WS_WB = XB_BYTES;
constexpr size_t WB_BYTES = (size_t)KD * NL * 2;
constexpr size_t WS_WFLAG = WS_WB + WB_BYTES;
constexpr size_t WS_XFLAG = WS_WFLAG + 16384;
constexpr size_t WS_GMAX = WS_XFLAG + 8192;
constexpr size_t WS_LMAX = WS_GMAX + 4096;
constexpr size_t WS_HINT = WS_LMAX + 4096;
constexpr size_t WS_CTR = WS_HINT + 16384;
constexpr size_t WS_TMO = WS_CTR + 256;
constexpr size_t WS_DBG = WS_TMO + 256 + 3584;
constexpr size_t WS_END = WS_DBG + 65536 + 256 * 2048;

constexpr u64 DEADLINE = 100000000ull;
constexpr u64 HINT_WAIT = 10000ull;

struct Params {
  const float* x;
  char* ws[NR];
  const float* w;
  signed char* out;
  int rank;
};

typedef __attribute__((address_space(3))) unsigned lds_u32;

__device__ __forceinline__ __amdgpu_buffer_rsrc_t mk_rsrc(const void* p, unsigned bytes) {
  return __builtin_amdgcn_make_buffer_rsrc((void*)p, 0, bytes, 0x00020000);
}
__device__ __forceinline__ unsigned pk_bf16(float a, float b) {
  f32x2 v = {a, b};
  return __builtin_bit_cast(unsigned, __builtin_convertvector(v, bf16x2));
}
__device__ __forceinline__ void bar() { asm volatile("s_barrier" ::: "memory"); }
__device__ __forceinline__ void fence_c() { asm volatile("" ::: "memory"); }
template <int N> __device__ __forceinline__ void wait_vm() {
  asm volatile("s_waitcnt vmcnt(%0)" ::"i"(N) : "memory");
}
__device__ __forceinline__ u64 now() { return __builtin_amdgcn_s_memrealtime(); }
struct Tmo { unsigned* word; volatile lds_u32* mine; };
__device__ __forceinline__ void give_up(Tmo tmo, unsigned call) {
  *tmo.mine = 0xffffffffu;
  __hip_atomic_store(tmo.word, call, __ATOMIC_RELAXED, __HIP_MEMORY_SCOPE_AGENT);
}
__device__ __forceinline__ unsigned wave_max(unsigned v) {
#define DPP_MAX(c, m) v = max(v, (unsigned)__builtin_amdgcn_update_dpp(0, (int)v, c, m, 0xf, false))
  DPP_MAX(0x111, 0xf); DPP_MAX(0x112, 0xf); DPP_MAX(0x114, 0xf); DPP_MAX(0x118, 0xf);
  DPP_MAX(0x142, 0xa); DPP_MAX(0x143, 0xc);
  return (unsigned)__builtin_amdgcn_readlane((int)v, 63);
}
__device__ __forceinline__ bool timed_out(Tmo tmo, unsigned call, u64 t0, unsigned& spins) {
  if ((++spins & 31u) != 0u) return false;
  if (__hip_atomic_load(tmo.word, __ATOMIC_RELAXED, __HIP_MEMORY_SCOPE_AGENT) == call || now() - t0 > DEADLINE) {
    give_up(tmo, call);
    return true;
  }
  return false;
}
__device__ __forceinline__ void lds_wait(volatile lds_u32* w, unsigned need, Tmo tmo, unsigned call) {
  if (__all((int)(*w - need) >= 0)) return;
  const u64 t0 = now();
  unsigned spins = 0;
  while (!__all((int)(*w - need) >= 0)) {
    __builtin_amdgcn_s_sleep(1);
    if ((++spins & 1023u) == 0u && now() - t0 > 2 * DEADLINE) { give_up(tmo, call); break; }
  }
}
__device__ __forceinline__ void wait_src(char* myws, int src, int j, Tmo tmo, unsigned call) {
  u64* f = (u64*)(myws + WS_XFLAG) + src * NTILE + j;
  const u64 t0 = now();
  unsigned spins = 0;
  for (;;) {
    const u64 v = __hip_atomic_load(f, __ATOMIC_RELAXED, __HIP_MEMORY_SCOPE_SYSTEM);
    if (__all((unsigned)v >= call && (unsigned)(v >> 32) >= call)) break;
    if (timed_out(tmo, call, t0, spins)) break;
    __builtin_amdgcn_s_sleep(1);
  }
}

__device__ __forceinline__ void cvt_block(const float* x, int j, int mb, int lane, u32x4& v0, u32x4& v1) {
  const float* s = x + (size_t)((j & 1) * BM + mb * 16 + (lane & 15)) * KD + (j >> 1) * BK + (lane >> 4) * 8;
  const f32x4 a0 = *(const f32x4*)(s), a1 = *(const f32x4*)(s + 4);
  const f32x4 c0 = *(const f32x4*)(s + 32), c1 = *(const f32x4*)(s + 36);
  v0[0] = pk_bf16(a0[0], a0[1]); v0[1] = pk_bf16(a0[2], a0[3]);
  v0[2] = pk_bf16(a1[0], a1[1]); v0[3] = pk_bf16(a1[2], a1[3]);
  v1[0] = pk_bf16(c0[0], c0[1]); v1[1] = pk_bf16(c0[2], c0[3]);
  v1[2] = pk_bf16(c1[0], c1[1]); v1[3] = pk_bf16(c1[2], c1[3]);
}

#define STAMP(i) do {} while (0)

__global__ void __launch_bounds__(NTHREADS) ag_gemm_q8(Params p) {
  __shared__ __attribute__((aligned(1024))) char smem[LDS_BYTES];
  const int tid = threadIdx.x;
  const int lane = tid & 63;
  const int wv = __builtin_amdgcn_readfirstlane(tid >> 6);
  const int b = blockIdx.x;
  const int rank = p.rank;
  const int grp = b & 7, gi = b >> 3;
  const int src = (rank + grp) & 7;
  const int mt = gi >> 4, nt = gi & 15;
  char* const myws = p.ws[0];
  unsigned* const hint = (unsigned*)(myws + WS_HINT) + grp * 512;
  char* const sB = smem + NA * A_STAGE;
  volatile lds_u32* const ctl = (volatile lds_u32*)(smem + LDS_CTL);
  const Tmo tmo = {(unsigned*)(myws + WS_TMO), ctl + 16};

  if (wv < 8) {
    const int pj = b >> 1, mb = (b & 1) * 8 + wv;
    u32x4 v0, v1;
    cvt_block(p.x, pj, mb, lane, v0, v1);
    const __amdgpu_buffer_rsrc_t rx = mk_rsrc(myws + WS_XB, (unsigned)XB_BYTES);
    const int o = pj * A_STAGE + mb * 2048 + lane * 16;
    __builtin_amdgcn_raw_buffer_store_b128(v0, rx, o, 0, 17);
    __builtin_amdgcn_raw_buffer_store_b128(v1, rx, o + 1024, 0, 17);
    if (wv == 0) STAMP(7);
    wait_vm<0>();
    bar();
  }
  fence_c();
  if (wv == 8) STAMP(0);
  const unsigned ctr = __hip_atomic_load((unsigned*)(myws + WS_CTR), __ATOMIC_RELAXED, __HIP_MEMORY_SCOPE_AGENT);
#define GET_CALL() const unsigned call = __builtin_amdgcn_readfirstlane(ctr) + 1u

  if (wv >= 8 && wv < 12) {
    const int pw = wv - 8, cn = b & 15;
    const int cg = lane & 31, oct = pw * 2 + (lane >> 5);
    const __amdgpu_buffer_rsrc_t rw = mk_rsrc(p.w, (unsigned)((size_t)KD * NL * 4));
    const __amdgpu_buffer_rsrc_t rb = mk_rsrc(myws + WS_WB, (unsigned)WB_BYTES);
    const __amdgpu_buffer_rsrc_t rf = mk_rsrc(myws + WS_WFLAG, 16384);
    const int voff = oct * 8 * (NL * 4) + cn * (BN * 4) + cg * 16;
    const int wofs = (cg >> 4) * 8192 + (oct >> 2) * 1024 + ((oct & 3) * 16 + (cg & 15)) * 16;
    f32x4 R[8];
    const u64 t0 = now();
    bar();
    GET_CALL();
    int W = pw == 0 ? 0 : NKT;
    unsigned spins = 0;
    for (int kt = b >> 4; ; kt += 16) {
      const int f = kt < 8 ? NUNIT : max((kt >> 4) * 2 - 1, 0) + mt;
      while (kt < NKT ? now() - t0 < (u64)(kt * WPACE) && (int)ctl[11] < f : W < NKT) {
        if (W < NKT && W - (int)ctl[0] <= 6) {
          const u32x4 v = __builtin_amdgcn_raw_buffer_load_b128(rf, lane * 16, nt * (NKT * 16), 17);
          const u64 m = ~__ballot(v[0] == call && v[1] == call && v[2] == call && v[3] == call);
          W = m ? __builtin_ctzll(m) : NKT;
          if (timed_out(tmo, call, t0, spins)) W = NKT;
          ctl[9] = W;
        }
        __builtin_amdgcn_s_sleep(4);
      }
      if (kt >= NKT) break;
#pragma unroll
      for (int r = 0; r < 8; ++r)
        R[r] = __builtin_bit_cast(f32x4, __builtin_amdgcn_raw_buffer_load_b128(rw, voff, kt * (BK * NL * 4) + r * (NL * 4), WAUX));
#pragma unroll
      for (int t = 0; t < 4; ++t) {
        u32x4 v;
        v[0] = pk_bf16(R[0][t], R[1][t]); v[1] = pk_bf16(R[2][t], R[3][t]);
        v[2] = pk_bf16(R[4][t], R[5][t]); v[3] = pk_bf16(R[6][t], R[7][t]);
        __builtin_amdgcn_raw_buffer_store_b128(v, rb, wofs + t * 2048, (kt * 16 + cn) * B_STAGE, 17);
      }
      wait_vm<0>();
      if (lane == 0) __builtin_amdgcn_raw_buffer_store_b32(call, rf, pw * 4, (cn * NKT + kt) * 16, 17);
      if (pw == 0) TR(6, 32 + (kt >> 4));
    }
    return;
  }

  if (wv == 15) {
    GET_CALL();
    unsigned* xf = (unsigned*)(p.ws[lane & 7] + WS_XFLAG) + rank * NWG + b;
    asm volatile("" : "+v"(xf));
    bar();
    if (lane < NR) __hip_atomic_store(xf, call, __ATOMIC_RELAXED, __HIP_MEMORY_SCOPE_SYSTEM);
    STAMP(1);
    const __amdgpu_buffer_rsrc_t rb = mk_rsrc(myws + WS_WB, (unsigned)WB_BYTES);
    volatile lds_u32* const look = ctl + (lane < 8 ? lane : 9);
    const unsigned slack = lane < 8 ? NB : 0;
    int I = 0, S = 0, si = 0;
    u64 t0 = now();
    unsigned spins = 0;
    while (S < NKT) {
      if (I < NKT && I - S < 3 && __all((int)(*look + slack - (unsigned)(I + 1)) >= 0)) {
        const int so = (I * 16 + nt) * B_STAGE;
#pragma unroll
        for (int c = 0; c < 4; ++c) {
          __attribute__((address_space(3))) void* d =
              (__attribute__((address_space(3))) void*)(sB + si * B_STAGE + c * 4096);
          __builtin_amdgcn_raw_ptr_buffer_load_lds(rb, d, 16, lane * 16, so + c * 4096, 0, 0);
          __builtin_amdgcn_raw_ptr_buffer_load_lds(rb, d, 16, lane * 16, so + c * 4096, 1024, 0);
          __builtin_amdgcn_raw_ptr_buffer_load_lds(rb, d, 16, lane * 16, so + c * 4096, 2048, 0);
          __builtin_amdgcn_raw_ptr_buffer_load_lds(rb, d, 16, lane * 16, so + c * 4096, 3072, 0);
        }
        ++I;
        si = si == NB - 1 ? 0 : si + 1;
      } else if (S < I) {
        if (I - S == 3) wait_vm<32>(); else if (I - S == 2) wait_vm<16>(); else wait_vm<0>();
        TR(5, S);
        ++S;
        ctl[8] = S;
        t0 = now();
      } else {
        __builtin_amdgcn_s_sleep(1);
        if ((++spins & 1023u) == 0u && now() - t0 > 2 * DEADLINE) { give_up(tmo, call); ctl[8] = NKT; break; }
      }
    }
    return;
  }

  if (wv == 12) {
    if (lane < 32) ctl[lane] = 0;
    if (gi == 0) asm volatile("buffer_inv sc0 sc1" ::: "memory");
    asm volatile("s_waitcnt vmcnt(0) lgkmcnt(0)" ::: "memory");
    bar();
    GET_CALL();
    const __amdgpu_buffer_rsrc_t ra = mk_rsrc(p.ws[grp] + WS_XB, (unsigned)XB_BYTES);
    const int avoff = lane * 16;
    volatile lds_u32* const look = ctl + (lane < 8 ? lane : 13);
    const unsigned slack = lane < 8 ? NA : 0;
    int I = 0, S = 0, si = 0;
    u64 t0 = now();
    unsigned spins = 0;
    while (S < NKT) {
      if (I < NKT && I - S < 2 && __all((int)(*look + slack - (unsigned)(I + 1)) >= 0)) {
        const int so = (I * 2 + mt) * A_STAGE;
#pragma unroll
        for (int c = 0; c < 8; ++c) {
          __attribute__((address_space(3))) void* d =
              (__attribute__((address_space(3))) void*)(smem + si * A_STAGE + c * 4096);
          __builtin_amdgcn_raw_ptr_buffer_load_lds(ra, d, 16, avoff, so + c * 4096, 0, 0);
          __builtin_amdgcn_raw_ptr_buffer_load_lds(ra, d, 16, avoff, so + c * 4096, 1024, 0);
          __builtin_amdgcn_raw_ptr_buffer_load_lds(ra, d, 16, avoff, so + c * 4096, 2048, 0);
          __builtin_amdgcn_raw_ptr_buffer_load_lds(ra, d, 16, avoff, so + c * 4096, 3072, 0);
        }
        TR(1, I);
        ++I;
        si = si == NA - 1 ? 0 : si + 1;
      } else if (S < I) {
        if (I - S == 2) wait_vm<32>(); else wait_vm<0>();
        TR(2, S);
        ++S;
        ctl[12] = S;
        t0 = now();
      } else {
        __builtin_amdgcn_s_sleep(1);
        if ((++spins & 1023u) == 0u && now() - t0 > 2 * DEADLINE) { give_up(tmo, call); ctl[12] = NKT; break; }
      }
    }
    return;
  }

  if (wv == 13) {
    bar();
    GET_CALL();
    const __amdgpu_buffer_rsrc_t ra = mk_rsrc(p.ws[grp] + WS_XB, (unsigned)XB_BYTES);
    const __amdgpu_buffer_rsrc_t rh = mk_rsrc(hint, NUNIT * 4);
    for (int q = gi ^ 16; q < NUNIT; q += 32) {
      wait_src(myws, src, q / UPT, tmo, call);
      const u64 t0 = now();
      unsigned spins = 0;
      for (;;) {
        const unsigned ph = ctl[14];
        if (ph != 0u && (int)((unsigned)now() - ph - (unsigned)(q * TAU / 16)) >= 0 &&
            q / (2 * UPT) - (int)ctl[0] <= AHEAD) break;
        __builtin_amdgcn_s_sleep(1);
        if ((++spins & 255u) == 0u && now() - t0 > 2 * HINT_WAIT) break;
      }
      TR(6, q >> 5);
#pragma unroll
      for (int u = 0; u < 4 / UPT; ++u)
        __builtin_amdgcn_raw_ptr_buffer_load_lds(ra, (__attribute__((address_space(3))) void*)(smem + LDS_CTL + 256),
                                                 4, lane * 128, q * (A_STAGE / UPT) + u * 8192, 0, 0);
      wait_vm<0>();
      if (lane == 0) __builtin_amdgcn_raw_buffer_store_b32(call, rh, q * 4, 0, 0);
      ctl[11] = (q >> 5) + 1;
      TR(6, 16 + (q >> 5));
    }
    return;
  }

  if (wv == 14) {
    bar();
    GET_CALL();
    const __amdgpu_buffer_rsrc_t rh = mk_rsrc(hint, NUNIT * 4);
    wait_src(myws, src, 0, tmo, call);
    const unsigned base = (unsigned)now();
    int ph = -BACKLOG * TAU;
    ctl[14] = (base + ph / 16) | 1u;
    int R = 0, strikes = 0;
    u64 t0 = now();
    unsigned spins = 0;
    while (R < NKT) {
      fence_c();
      const unsigned ts = (unsigned)now();
      const u32x4 v = __builtin_amdgcn_raw_buffer_load_b128(rh, lane * 16, 0, HAUX);
      const u64 m0 = __ballot(v[0] == call), m1 = __ballot(v[1] == call);
      const u64 m2 = __ballot(v[2] == call), m3 = __ballot(v[3] == call);
      const u64 all = ~(mt ? m2 & m3 : m0 & m1);
      const int n = __builtin_popcountll(m0) + __builtin_popcountll(m1) + __builtin_popcountll(m2) + __builtin_popcountll(m3);
      const int r = all ? __builtin_ctzll(all) : NKT;
      if (n > 0) {
        const int el = min((int)(ts - base), 1 << 24);
        const int d = (el - LEAD) * 16 - n * TAU - ph;
        ph += d > 0 ? d >> SMOOTH : d >> SMOOTHD;
        ctl[14] = (base + ph / 16) | 1u;
      }
      if (r > R) {
        R = r; ctl[13] = R;
        t0 = now(); strikes = 0;
      } else if ((++spins & 63u) == 0u && now() - t0 > HINT_WAIT) {
        wait_src(myws, src, R * 2 + mt, tmo, call);
        if (++strikes == 2) break;
        t0 = now();
      }
    }
    for (; R < NKT; ++R) {
      wait_src(myws, src, R * 2 + mt, tmo, call);
      ctl[13] = R + 1;
    }
    return;
  }

  char* peer = p.ws[lane & 7];
  asm volatile("" : "+v"(peer));
  GET_CALL();

  const int wm = wv >> 1, wn = wv & 1;
  f32x4 acc[4][4];
#pragma unroll
  for (int i = 0; i < 4; ++i)
#pragma unroll
    for (int j = 0; j < 4; ++j) acc[i][j] = (f32x4){0.f, 0.f, 0.f, 0.f};

  const char* const pa0 = smem + wm * 8192 + lane * 16;
  const char* const pb0 = smem + NA * A_STAGE + wn * 8192 + lane * 16;
  volatile lds_u32* const rdy = ctl + 8 + (lane & 4);
  int sa = 0, sb = 0;
  for (int t = 0; t < NKT; ++t) {
    lds_wait(rdy, t + 1, tmo, call);
    if (wv == 0) TR(3, t);
    fence_c();
    const char* pa = pa0 + sa * A_STAGE;
    const char* pb = pb0 + sb * B_STAGE;
    bf16x8 w0[4], w1[4], x0[4], x1[4];
#pragma unroll
    for (int i = 0; i < 4; ++i) w0[i] = *(const bf16x8*)(pb + i * 2048);
#pragma unroll
    for (int i = 0; i < 4; ++i) x0[i] = *(const bf16x8*)(pa + i * 2048);
#pragma unroll
    for (int i = 0; i < 4; ++i) w1[i] = *(const bf16x8*)(pb + i * 2048 + 1024);
    __builtin_amdgcn_sched_barrier(0);
#pragma unroll
    for (int i = 0; i < 4; ++i) {
#pragma unroll
      for (int j = 0; j < 4; ++j)
        acc[i][j] = __builtin_amdgcn_mfma_f32_16x16x32_bf16(w0[j], x0[i], acc[i][j], 0, 0, 0);
      __builtin_amdgcn_sched_barrier(0);
      x1[i] = *(const bf16x8*)(pa + i * 2048 + 1024);
      __builtin_amdgcn_sched_barrier(0);
    }
    fence_c();
    ctl[wv] = t + 1;
    fence_c();
    __builtin_amdgcn_sched_barrier(0);
#pragma unroll
    for (int i = 0; i < 4; ++i)
#pragma unroll
      for (int j = 0; j < 4; ++j)
        acc[i][j] = __builtin_amdgcn_mfma_f32_16x16x32_bf16(w1[j], x1[i], acc[i][j], 0, 0, 0);
    __builtin_amdgcn_sched_barrier(0);
    if (wv == 0) TR(4, t);
    sa = sa == NA - 1 ? 0 : sa + 1;
    sb = sb == NB - 1 ? 0 : sb + 1;
  }
  if (wv == 0) STAMP(3);

  float m = 0.f;
#pragma unroll
  for (int i = 0; i < 4; ++i)
#pragma unroll
    for (int j = 0; j < 4; ++j)
#pragma unroll
      for (int r = 0; r < 4; ++r) m = fmaxf(m, fabsf(acc[i][j][r]));
  unsigned mu = wave_max(__builtin_bit_cast(unsigned, m));
  unsigned in = 0;
  if (lane == 0) {
    __hip_atomic_fetch_max((lds_u32*)(ctl + 16), mu, __ATOMIC_RELAXED, __HIP_MEMORY_SCOPE_WORKGROUP);
    fence_c();
    in = __hip_atomic_fetch_add((lds_u32*)(ctl + 17), 1u, __ATOMIC_RELAXED, __HIP_MEMORY_SCOPE_WORKGROUP);
  }
  fence_c();
  mu = ctl[16];
  if (__builtin_amdgcn_readfirstlane(in) == 7u) {
    const u64 t0 = now();
    unsigned spins = 1;
    const __amdgpu_buffer_rsrc_t rl = mk_rsrc(myws + WS_LMAX, 4096);
    if (gi != 0) {
      const u32x2 v = {mu, call};
      if (lane == 0) {
        __builtin_amdgcn_raw_buffer_store_b64(v, rl, gi * 8, grp * 256, 0);
        __builtin_amdgcn_raw_buffer_store_b64(v, rl, gi * 8, grp * 256 + 2048, 16);
      }
    } else {
      for (;;) {
        fence_c();
        u32x2 v;
        if (spins & 7u) v = __builtin_amdgcn_raw_buffer_load_b64(rl, (lane & 31) * 8, grp * 256, HAUX);
        else v = __builtin_amdgcn_raw_buffer_load_b64(rl, (lane & 31) * 8, grp * 256 + 2048, 16);
        if ((lane & 31) == 0) { v[0] = 0u; v[1] = call; }
        if (__all(v[1] == call)) { mu = max(mu, v[0]); break; }
        if (timed_out(tmo, call, t0, spins)) { mu = 0xffffffffu; break; }
      }
      mu = wave_max(mu);
      __hip_atomic_store((u64*)(peer + WS_GMAX) + (lane >> 3) * 64 + rank * 8 + grp, ((u64)call << 32) | mu,
                         __ATOMIC_RELAXED, __HIP_MEMORY_SCOPE_SYSTEM);
    }
    STAMP(4);
    u64* gm = (u64*)(myws + WS_GMAX) + grp * 64 + lane;
    for (;;) {
      const u64 v = __hip_atomic_load(gm, __ATOMIC_RELAXED, __HIP_MEMORY_SCOPE_SYSTEM);
      if (__all((unsigned)(v >> 32) == call)) { mu = (unsigned)v; break; }
      if (timed_out(tmo, call, t0, spins)) { mu = 0xffffffffu; break; }
    }
    mu = wave_max(mu);
    if (lane == 0) { ctl[18] = mu; ctl[19] = 1u; }
    if (b == 0 && lane == 0)
      __hip_atomic_store((unsigned*)(myws + WS_CTR), call, __ATOMIC_RELAXED, __HIP_MEMORY_SCOPE_AGENT);
    STAMP(5);
  }
  lds_wait(ctl + 19, 1, tmo, call);
  mu = ctl[18];
  const bool bad = mu == 0xffffffffu || ctl[19] != 1u;
  const float amax = __builtin_bit_cast(float, mu);
  const float inv = amax > 0.f ? 127.0f / amax : 0.f;

  const __amdgpu_buffer_rsrc_t ro = mk_rsrc(p.out, NR * MS * NL);
  const int oo = (src * MS + mt * BM + wm * 64 + (lane & 15)) * NL + nt * BN + wn * 64 + (lane >> 4) * 16;
#pragma unroll
  for (int i = 0; i < 4; ++i) {
    u32x4 v;
#pragma unroll
    for (int r = 0; r < 4; ++r) {
      unsigned q[4];
#pragma unroll
      for (int j = 0; j < 4; ++j) q[j] = __builtin_bit_cast(unsigned, __builtin_fmaf(acc[i][j][r], inv, 12582912.0f));
      const unsigned d = __builtin_amdgcn_perm(__builtin_amdgcn_perm(q[3], q[2], 0x0c0c0400u),
                                               __builtin_amdgcn_perm(q[1], q[0], 0x0c0c0400u), 0x05040100u);
      v[r] = bad ? 0x80808080u : d;
    }
    __builtin_amdgcn_raw_buffer_store_b128(v, ro, oo, i * 16 * NL, OAUX);
  }
  if (wv == 0) STAMP(6);
}

}

extern "C" void kernel_launch(int rank, int nranks, ncclComm_t comm, hipStream_t stream,
                              void* const* d_in, const int* in_sizes, int n_in, void* d_out,
                              int out_size, const int* in_dtypes, int out_dtype,
                              void* const* d_out_all, void* const* d_ws_all, size_t ws_bytes) {
  (void)comm; (void)in_dtypes; (void)out_dtype; (void)d_out_all;
  if (nranks != NR || (unsigned)rank >= (unsigned)NR || n_in < 2 || ws_bytes < WS_END ||
      in_sizes[0] != MS * KD || in_sizes[1] != KD * NL || out_size != NR * MS * NL) {
    fprintf(stderr, "unexpected problem shape\n");
    return;
  }
  Params p;
  p.x = (const float*)d_in[0];
  p.w = (const float*)d_in[1];
  p.out = (signed char*)d_out;
  for (int i = 0; i < NR; ++i) p.ws[i] = (char*)d_ws_all[(rank + i) % NR];
  p.rank = rank;
  hipLaunchKernelGGL(ag_gemm_q8, dim3(NWG), dim3(NTHREADS), 0, stream, p);
  HIP_CHECK(hipGetLastError());
}
```

```cpp
#include <hip/hip_runtime.h>
#include <rccl/rccl.h>
#include <cstdio>

#define HIP_CHECK(expr) do { hipError_t _e = (expr); if (_e != hipSuccess) { \
    fprintf(stderr, "HIP error %s at %s:%d\n", hipGetErrorString(_e), __FILE__, __LINE__); \
    return; } } while (0)

namespace {

typedef __attribute__((ext_vector_type(4))) unsigned u32x4;
typedef __attribute__((ext_vector_type(2))) unsigned u32x2;
typedef __attribute__((ext_vector_type(4))) float f32x4;
typedef __attribute__((ext_vector_type(2))) float f32x2;
typedef __attribute__((ext_vector_type(2))) __bf16 bf16x2;
typedef __attribute__((ext_vector_type(8))) __bf16 bf16x8;
typedef unsigned long long u64;

constexpr int NR = 8;
constexpr int KD = 4096;
constexpr int NL = 2048;
constexpr int MS = 512;
constexpr int BM = 256, BN = 128, BK = 64;
constexpr int NKT = KD / BK;
constexpr int NTILE = 2 * NKT;
constexpr int NUNIT = 2 * NTILE;
constexpr int NA = 3, NB = 3;
constexpr int NG = 26, NC = 4;
#define FD 7
#define EARLY 9
#define XPACE 25
#define SW 11
#define ROT 1
#define SKIP 3
#define AHEAD 5
#define WPACE 60
constexpr int HAUX = 3;
constexpr int A_STAGE = BM * BK * 2;
constexpr int W_TILE = BN * BK * 2;
constexpr int B_STAGE = W_TILE + 4096;
constexpr int LDS_CTL = NA * A_STAGE + NB * B_STAGE;
constexpr int LDS_BYTES = LDS_CTL + 512;
#define TR(k, i) do {} while (0)
constexpr int NWG = 256;
constexpr int NTHREADS = 768;

constexpr size_t WS_XB = 0;
constexpr size_t XB_BYTES = (size_t)MS * KD * 2;
constexpr size_t WS_WB = XB_BYTES;
constexpr size_t WB_BYTES = (size_t)KD * NL * 2;
constexpr size_t WS_WFLAG = WS_WB + WB_BYTES;
constexpr size_t WS_XFLAG = WS_WFLAG + 16384;
constexpr size_t WS_GMAX = WS_XFLAG + 8192;
constexpr size_t WS_LMAX = WS_GMAX + 4096;
constexpr size_t WS_HINT = WS_LMAX + 4096;
constexpr size_t WS_CTR = WS_HINT + 16384;
constexpr size_t WS_TMO = WS_CTR + 256;
constexpr size_t WS_DBG = WS_TMO + 256 + 3584;
constexpr size_t WS_END = WS_DBG + 65536 + 256 * 2048;

constexpr u64 DEADLINE = 100000000ull;
constexpr u64 HINT_WAIT = 10000ull;

struct Params {
  const float* x;
  char* ws[NR];
  const float* w;
  signed char* out;
  int rank;
};

typedef __attribute__((address_space(3))) unsigned lds_u32;

__device__ __forceinline__ __amdgpu_buffer_rsrc_t mk_rsrc(const void* p, unsigned bytes) {
  return __builtin_amdgcn_make_buffer_rsrc((void*)p, 0, bytes, 0x00020000);
}
__device__ __forceinline__ unsigned pk_bf16(float a, float b) {
  f32x2 v = {a, b};
  return __builtin_bit_cast(unsigned, __builtin_convertvector(v, bf16x2));
}
__device__ __forceinline__ void bar() { asm volatile("s_barrier" ::: "memory"); }
__device__ __forceinline__ void fence_c() { asm volatile("" ::: "memory"); }
template <int N> __device__ __forceinline__ void wait_vm() {
  asm volatile("s_waitcnt vmcnt(%0)" ::"i"(N) : "memory");
}
__device__ __forceinline__ u64 now() { return __builtin_amdgcn_s_memrealtime(); }
struct Tmo { unsigned* word; volatile lds_u32* mine; };
__device__ __forceinline__ void give_up(Tmo tmo, unsigned call) {
  *tmo.mine = 0xffffffffu;
  __hip_atomic_store(tmo.word, call, __ATOMIC_RELAXED, __HIP_MEMORY_SCOPE_AGENT);
}
__device__ __forceinline__ unsigned wave_max(unsigned v) {
#define DPP_MAX(c, m) v = max(v, (unsigned)__builtin_amdgcn_update_dpp(0, (int)v, c, m, 0xf, false))
  DPP_MAX(0x111, 0xf); DPP_MAX(0x112, 0xf); DPP_MAX(0x114, 0xf); DPP_MAX(0x118, 0xf);
  DPP_MAX(0x142, 0xa); DPP_MAX(0x143, 0xc);
  return (unsigned)__builtin_amdgcn_readlane((int)v, 63);
}
__device__ __forceinline__ bool timed_out(Tmo tmo, unsigned call, u64 t0, unsigned& spins) {
  if ((++spins & 31u) != 0u) return false;
  if (__hip_atomic_load(tmo.word, __ATOMIC_RELAXED, __HIP_MEMORY_SCOPE_AGENT) == call || now() - t0 > DEADLINE) {
    give_up(tmo, call);
    return true;
  }
  return false;
}
__device__ __forceinline__ void lds_wait(volatile lds_u32* w, unsigned need, Tmo tmo, unsigned call) {
  if (__all((int)(*w - need) >= 0)) return;
  const u64 t0 = now();
  unsigned spins = 0;
  while (!__all((int)(*w - need) >= 0)) {
    __builtin_amdgcn_s_sleep(1);
    if ((++spins & 1023u) == 0u && now() - t0 > 2 * DEADLINE) { give_up(tmo, call); break; }
  }
}
__device__ __forceinline__ void wait_src(char* myws, int src, int j, Tmo tmo, unsigned call) {
  u64* f = (u64*)(myws + WS_XFLAG) + src * NTILE + j;
  const u64 t0 = now();
  unsigned spins = 0;
  for (;;) {
    const u64 v = __hip_atomic_load(f, __ATOMIC_RELAXED, __HIP_MEMORY_SCOPE_SYSTEM);
    if (__all((unsigned)v >= call && (unsigned)(v >> 32) >= call)) break;
    if (timed_out(tmo, call, t0, spins)) break;
    __builtin_amdgcn_s_sleep(1);
  }
}

__device__ __forceinline__ void cvt_block(const float* x, int j, int mb, int lane, u32x4& v0, u32x4& v1) {
  const float* s = x + (size_t)((j & 1) * BM + mb * 16 + (lane & 15)) * KD + (j >> 1) * BK + (lane >> 4) * 8;
  const f32x4 a0 = *(const f32x4*)(s), a1 = *(const f32x4*)(s + 4);
  const f32x4 c0 = *(const f32x4*)(s + 32), c1 = *(const f32x4*)(s + 36);
  v0[0] = pk_bf16(a0[0], a0[1]); v0[1] = pk_bf16(a0[2], a0[3]);
  v0[2] = pk_bf16(a1[0], a1[1]); v0[3] = pk_bf16(a1[2], a1[3]);
  v1[0] = pk_bf16(c0[0], c0[1]); v1[1] = pk_bf16(c0[2], c0[3]);
  v1[2] = pk_bf16(c1[0], c1[1]); v1[3] = pk_bf16(c1[2], c1[3]);
}

#define STAMP(i) do {} while (0)

typedef __attribute__((address_space(3))) void* lds_ptr;
#define DMA4K(r, d, so) do { \
    __builtin_amdgcn_raw_ptr_buffer_load_lds(r, (lds_ptr)(d), 16, lane * 16, so, 0, 0); \
    __builtin_amdgcn_raw_ptr_buffer_load_lds(r, (lds_ptr)(d), 16, lane * 16, so, 1024, 0); \
    __builtin_amdgcn_raw_ptr_buffer_load_lds(r, (lds_ptr)(d), 16, lane * 16, so, 2048, 0); \
    __builtin_amdgcn_raw_ptr_buffer_load_lds(r, (lds_ptr)(d), 16, lane * 16, so, 3072, 0); } while (0)

__global__ void __launch_bounds__(NTHREADS) ag_gemm_q8(Params p) {
  __shared__ __attribute__((aligned(1024))) char smem[LDS_BYTES];
  const int tid = threadIdx.x;
  const int lane = tid & 63;
  const int wv = __builtin_amdgcn_readfirstlane(tid >> 6);
  const int b = blockIdx.x;
  const int rank = p.rank;
  const int grp = b & 7, gi = b >> 3;
  const int src = (rank + grp) & 7;
  const int mt = gi >= NG / 2, nt = gi - mt * (NG / 2);
  char* const myws = p.ws[0];
  char* const sB = smem + NA * A_STAGE;
  volatile lds_u32* const ctl = (volatile lds_u32*)(smem + LDS_CTL);
  const Tmo tmo = {(unsigned*)(myws + WS_TMO), ctl + 16};
  const __amdgpu_buffer_rsrc_t rh = mk_rsrc(myws + WS_HINT + grp * 2048, NUNIT * 4);
  const __amdgpu_buffer_rsrc_t ra = mk_rsrc(p.ws[grp] + WS_XB, (unsigned)XB_BYTES);
  const __amdgpu_buffer_rsrc_t rb = mk_rsrc(myws + WS_WB, (unsigned)WB_BYTES);
  const __amdgpu_buffer_rsrc_t rf = mk_rsrc(myws + WS_WFLAG, 16384);

  if (wv < 8) {
    const int pj = b >> 1, mb = (b & 1) * 8 + wv;
    u32x4 v0, v1;
    cvt_block(p.x, pj, mb, lane, v0, v1);
    const __amdgpu_buffer_rsrc_t rx = mk_rsrc(myws + WS_XB, (unsigned)XB_BYTES);
    const int o = pj * A_STAGE + mb * 2048 + lane * 16;
    __builtin_amdgcn_raw_buffer_store_b128(v0, rx, o, 0, 17);
    __builtin_amdgcn_raw_buffer_store_b128(v1, rx, o + 1024, 0, 17);
    if (wv == 0) STAMP(7);
    wait_vm<0>();
    bar();
  } else if (wv == 9) {
    const unsigned c = __hip_atomic_load((unsigned*)(myws + WS_CTR), __ATOMIC_RELAXED, __HIP_MEMORY_SCOPE_AGENT) + 1u;
    unsigned* xf = (unsigned*)(p.ws[lane & 7] + WS_XFLAG) + rank * NWG + b;
    asm volatile("" : "+v"(xf));
    if (lane < 32) ctl[lane] = lane == 15 ? c : 0u;
    asm volatile("s_waitcnt vmcnt(0) lgkmcnt(0)" ::: "memory");
    bar();
    if (lane < NR) __hip_atomic_store(xf, c, __ATOMIC_RELAXED, __HIP_MEMORY_SCOPE_SYSTEM);
    STAMP(1);
  } else {
    if (wv == 8) STAMP(0);
    if (wv == 8 && gi >= NG + NC) asm volatile("buffer_inv sc0 sc1\n s_waitcnt vmcnt(0)" ::: "memory");
    bar();
  }
  fence_c();
  const unsigned call = __builtin_amdgcn_readfirstlane(ctl[15]);
  const u64 ts = now();

  if (gi >= NG + NC) {
    const int half = gi - (NG + NC);
    if (wv == 0) {
      const __amdgpu_buffer_rsrc_t rs = mk_rsrc(myws + WS_XFLAG + src * 1024, 1024);
      int ready = 0, in = 0;
      unsigned spins = 0;
      for (int i = 0; i < NUNIT; ++i) {
        while (ready <= i >> 2) {
          fence_c();
          const u32x4 v = __builtin_amdgcn_raw_buffer_load_b128(rs, lane * 16, 0, 17);
          const u64 m = ~__ballot(v[0] >= call && v[1] >= call && v[2] >= call && v[3] >= call);
          ready = m ? __builtin_ctzll(m) : NKT;
          ctl[2] = in = max(in, i);
          if (timed_out(tmo, call, ts, spins)) return;
        }
        while (now() - ts < (u64)(i * XPACE)) __builtin_amdgcn_s_sleep(1);
        __builtin_amdgcn_raw_ptr_buffer_load_lds(ra, (lds_ptr)smem, 4, lane * 128,
                                                 (i >> 1) * A_STAGE + half * 16384 + (i & 1) * 8192, 0, 0);
        wait_vm<FD - 1>();
        in = max(in, min(i + 2 - FD + EARLY, ready * 4));
        ctl[2] = in;
      }
      if (FD - 2 >= EARLY + 2) { wait_vm<EARLY + 2>(); ctl[2] = NUNIT - 2; }
      wait_vm<EARLY>(); ctl[2] = NUNIT;
      wait_vm<0>();
      STAMP(3);
    } else if (wv == 1) {
      int done = 0;
      unsigned spins = 0;
      while (done < NTILE && ctl[16] != 0xffffffffu) {
        const int n = min((int)ctl[2] >> 1, done + 64);
        if (lane < n - done) __builtin_amdgcn_raw_buffer_store_b32(call, rh, (done + lane) * 8 + half * 4, 0, 0);
        done = max(done, n);
        if ((++spins & 1023u) == 0u && now() - ts > 2 * DEADLINE) break;
      }
    }
    return;
  }

  if (gi >= NG) {
    const int pw = wv & 3, cg = lane & 31, oct = pw * 2 + (lane >> 5);
    const __amdgpu_buffer_rsrc_t rw = mk_rsrc(p.w, (unsigned)((size_t)KD * NL * 4));
    const int voff = oct * 8 * (NL * 4) + cg * 16;
    const int wo0 = (cg >> 4) * 8192 + (oct >> 2) * 1024 + ((oct & 3) * 16 + (cg & 15)) * 16;
    const int wo1 = (lane >> 5) * 2048 + pw * 512 + cg * 16;
    char* const tp = smem + wv * 4096;
    for (int T = (wv >> 2) * (NC * 8) + (gi - NG) * 8 + grp; T < (wv < 8 ? NKT * 16 : min(SW * 96, NKT * 16)); T += T < SW * 96 ? 96 : 64) {
      const int kt = T >> 4, cn = T & 15;
      while (now() - ts < (u64)(kt * WPACE)) __builtin_amdgcn_s_sleep(2);
      f32x4 R[8];
#pragma unroll
      for (int r = 0; r < 8; ++r)
        R[r] = __builtin_bit_cast(f32x4, __builtin_amdgcn_raw_buffer_load_b128(
                   rw, voff, kt * (BK * NL * 4) + r * (NL * 4) + cn * (BN * 4), 2));
      u32x4 v[4];
#pragma unroll
      for (int t = 0; t < 4; ++t) {
        v[t][0] = pk_bf16(R[0][t], R[1][t]); v[t][1] = pk_bf16(R[2][t], R[3][t]);
        v[t][2] = pk_bf16(R[4][t], R[5][t]); v[t][3] = pk_bf16(R[6][t], R[7][t]);
      }
      if (cn >= NG / 2) {
#pragma unroll
        for (int t = 0; t < 4; ++t) *(u32x4*)(tp + (cg >> 2) * 512 + (lane >> 5) * 256 + (cg & 3) * 64 + t * 16) = v[t];
        fence_c();
#pragma unroll
        for (int t = 0; t < 4; ++t) v[t] = *(const u32x4*)(tp + t * 1024 + lane * 16);
        fence_c();
      }
      const int wo = cn < NG / 2 ? wo0 : wo1, ws = cn < NG / 2 ? 2048 : 4096;
#pragma unroll
      for (int t = 0; t < 4; ++t)
        __builtin_amdgcn_raw_buffer_store_b128(v[t], rb, wo, T * W_TILE + t * ws, 17);
      wait_vm<0>();
      if (lane == 0) __builtin_amdgcn_raw_buffer_store_b32(call, rf, pw * 4, (cn * NKT + kt) * 16, 17);
    }
    if (wv == 0) STAMP(3);
    if (wv < 8) return;
    const int cn = (gi - NG) * 4 + pw;
    int W = 0, kt = 0;
    u64 t0 = now();
    while (kt < NKT && now() - t0 < HINT_WAIT) {
      fence_c();
      if (W <= kt) {
        const u32x4 v = __builtin_amdgcn_raw_buffer_load_b128(rf, lane * 16, cn * (NKT * 16), 17);
        const u64 m = ~__ballot(v[0] == call && v[1] == call && v[2] == call && v[3] == call);
        W = m ? __builtin_ctzll(m) : NKT;
      }
      const u64 h = ~__ballot(__builtin_amdgcn_raw_buffer_load_b32(rh, lane * 16, 0, HAUX) == call);
      const int R = h ? __builtin_ctzll(h) : NKT;
      kt = max(kt, R + SKIP);
      for (; kt < W && kt <= R + AHEAD; ++kt) {
        __builtin_amdgcn_raw_ptr_buffer_load_lds(rb, (lds_ptr)tp, 4, lane * 128, (kt * 16 + cn) * W_TILE, 0, 0);
        __builtin_amdgcn_raw_ptr_buffer_load_lds(rb, (lds_ptr)tp, 4, lane * 128, (kt * 16 + cn) * W_TILE + 8192, 0, 0);
        t0 = now();
      }
      __builtin_amdgcn_s_sleep(2);
    }
    wait_vm<0>();
    return;
  }

  if (wv >= 8) __builtin_amdgcn_s_setprio(3);
  const int ex = nt == NG / 2 - 1 ? 0 : nt;
  if (wv == 8 || wv == 11) {
    const int odd = wv & 1;
    volatile lds_u32* const look = ctl + (lane < 8 ? lane : 13);
    const unsigned slack = lane < 8 ? NA : 0;
    int si = odd;
    for (int I = odd; I < NKT; I += 2) {
      lds_wait(look, I + 1 - slack, tmo, call);
      if (ctl[16] == 0xffffffffu) break;
#pragma unroll
      for (int c0 = 0; c0 < 8; ++c0) {
        const int c = (c0 + ROT * nt) & 7;
        DMA4K(ra, smem + si * A_STAGE + c * 4096, (I * 2 + mt) * A_STAGE + c * 4096);
      }
      TR(1, I);
      wait_vm<0>();
      ctl[12 + 2 * odd] = I + 1;
      TR(2, I);
      si = si == 0 ? 2 : si - 1;
    }
    ctl[12 + 2 * odd] = NKT;
    return;
  }

  if (wv == 9) {
    volatile lds_u32* const look = ctl + (lane < 8 ? lane : 9);
    const unsigned slack = lane < 8 ? NB : 0;
    int I = 0, S = 0, si = 0;
    u64 t0 = ts;
    unsigned spins = 0;
    while (S < NKT) {
      if (I < NKT && I - S < 3 && __all((int)(*look + slack - (unsigned)(I + 1)) >= 0)) {
#pragma unroll
        for (int c = 0; c < 4; ++c) DMA4K(rb, sB + si * B_STAGE + c * 4096, (I * 16 + nt) * W_TILE + c * 4096);
        DMA4K(rb, sB + si * B_STAGE + W_TILE, (I * 16 + NG / 2) * W_TILE + ex * 4096);
        ++I;
        si = si == 2 ? 0 : si + 1;
      } else if (S < I) {
        if (I - S == 3) wait_vm<40>(); else if (I - S == 2) wait_vm<20>(); else wait_vm<0>();
        TR(5, S);
        ++S;
        ctl[8] = S;
        t0 = now();
      } else {
        __builtin_amdgcn_s_sleep(1);
        if ((++spins & 1023u) == 0u && now() - t0 > 2 * DEADLINE) { give_up(tmo, call); ctl[8] = NKT; break; }
      }
    }
    return;
  }

  if (wv == 10) {
    int R = 0, W = 0, strikes = 0;
    u64 t0 = ts;
    unsigned spins = 0, sw = 0;
    while (R < NKT || W < NKT) {
      fence_c();
      if (W < NKT && W - (int)ctl[0] <= 6 && ((spins & 3u) == 0u || R >= W)) {
        const u32x4 v = __builtin_amdgcn_raw_buffer_load_b128(rf, lane * 16, nt * (NKT * 16), 17);
        const u32x4 e = __builtin_amdgcn_raw_buffer_load_b128(rf, lane * 16, (NG / 2 + (ex >> 2)) * (NKT * 16), 17);
        const u64 m = ~__ballot(v[0] == call && v[1] == call && v[2] == call && v[3] == call &&
                                e[0] == call && e[1] == call && e[2] == call && e[3] == call);
        W = m ? __builtin_ctzll(m) : NKT;
        if (timed_out(tmo, call, ts, sw)) W = NKT;
        ctl[9] = W;
      }
      ++spins;
      if (ctl[16] == 0xffffffffu) break;
      if (R == NKT) {
        __builtin_amdgcn_s_sleep(4);
        if (now() - ts > 2 * DEADLINE) give_up(tmo, call);
        continue;
      }
      int r = R;
      if (strikes < 2) {
        const u32x2 v = __builtin_amdgcn_raw_buffer_load_b64(rh, (R + (lane & 7)) * 16 + mt * 8, 0, HAUX);
        r += __builtin_ctz(~((unsigned)__ballot(v[0] == call && v[1] == call) & 255u));
      } else {
        wait_src(myws, src, R * 2 + mt, tmo, call);
        ++r;
      }
      if (r > R) {
        R = r; ctl[13] = R;
        t0 = now();
        if (strikes < 2) strikes = 0;
      } else if ((spins & 63u) == 0u && now() - t0 > HINT_WAIT) {
        wait_src(myws, src, R * 2 + mt, tmo, call);
        if (++strikes == 2) asm volatile("buffer_inv sc0 sc1\n s_waitcnt vmcnt(0)" ::: "memory");
        t0 = now();
      }
    }
    ctl[9] = NKT; ctl[13] = NKT;
    return;
  }

  char* peer = p.ws[lane & 7];
  asm volatile("" : "+v"(peer));
  const int wm = wv >> 1, wn = wv & 1;
  f32x4 acc[4][5];
#pragma unroll
  for (int i = 0; i < 4; ++i)
#pragma unroll
    for (int j = 0; j < 5; ++j) acc[i][j] = (f32x4){0.f, 0.f, 0.f, 0.f};

  const char* const pa0 = smem + wm * 8192 + lane * 16;
  const char* const pb0 = sB + wn * 8192 + lane * 16;
  const char* const pe0 = sB + W_TILE + wn * 2048 + lane * 16;
  volatile lds_u32* const rdy = ctl + 8 + (lane & 4);
  int st = 0;
  for (int t = 0; t < NKT; ++t) {
    lds_wait(rdy + (t & 1) * (lane & 4) / 2, t + 1, tmo, call);
    if (wv == 0) TR(3, t);
    fence_c();
    const char* pa = pa0 + st * A_STAGE;
    const char* pb = pb0 + st * B_STAGE;
    const char* pe = pe0 + st * B_STAGE;
    bf16x8 w0[5], w1[5], x0[4], x1[4];
#pragma unroll
    for (int i = 0; i < 4; ++i) w0[i] = *(const bf16x8*)(pb + i * 2048);
    w0[4] = *(const bf16x8*)pe;
#pragma unroll
    for (int i = 0; i < 4; ++i) x0[i] = *(const bf16x8*)(pa + i * 2048);
#pragma unroll
    for (int i = 0; i < 4; ++i) w1[i] = *(const bf16x8*)(pb + i * 2048 + 1024);
    w1[4] = *(const bf16x8*)(pe + 1024);
    __builtin_amdgcn_sched_barrier(0);
#pragma unroll
    for (int i = 0; i < 4; ++i) {
#pragma unroll
      for (int j = 0; j < 5; ++j)
        acc[i][j] = __builtin_amdgcn_mfma_f32_16x16x32_bf16(w0[j], x0[i], acc[i][j], 0, 0, 0);
      __builtin_amdgcn_sched_barrier(0);
      x1[i] = *(const bf16x8*)(pa + i * 2048 + 1024);
      __builtin_amdgcn_sched_barrier(0);
    }
    fence_c();
    ctl[wv] = t + 1;
    fence_c();
    __builtin_amdgcn_sched_barrier(0);
#pragma unroll
    for (int i = 0; i < 4; ++i)
#pragma unroll
      for (int j = 0; j < 5; ++j)
        acc[i][j] = __builtin_amdgcn_mfma_f32_16x16x32_bf16(w1[j], x1[i], acc[i][j], 0, 0, 0);
    __builtin_amdgcn_sched_barrier(0);
    if (wv == 0) TR(4, t);
    st = st == 2 ? 0 : st + 1;
  }
  if (wv == 0) STAMP(3);

  float m = 0.f;
#pragma unroll
  for (int i = 0; i < 4; ++i)
#pragma unroll
    for (int j = 0; j < 5; ++j)
#pragma unroll
      for (int r = 0; r < 4; ++r) m = fmaxf(m, fabsf(acc[i][j][r]));
  unsigned mu = wave_max(__builtin_bit_cast(unsigned, m));
  unsigned in = 0;
  if (lane == 0) {
    __hip_atomic_fetch_max((lds_u32*)(ctl + 16), mu, __ATOMIC_RELAXED, __HIP_MEMORY_SCOPE_WORKGROUP);
    fence_c();
    in = __hip_atomic_fetch_add((lds_u32*)(ctl + 17), 1u, __ATOMIC_RELAXED, __HIP_MEMORY_SCOPE_WORKGROUP);
  }
  fence_c();
  mu = ctl[16];
  if (__builtin_amdgcn_readfirstlane(in) == 7u) {
    const u64 t0 = now();
    unsigned spins = 1;
    const __amdgpu_buffer_rsrc_t rl = mk_rsrc(myws + WS_LMAX, 4096);
    if (gi != 0) {
      const u32x2 v = {mu, call};
      if (lane == 0) {
        __builtin_amdgcn_raw_buffer_store_b64(v, rl, gi * 8, grp * 256, 0);
        __builtin_amdgcn_raw_buffer_store_b64(v, rl, gi * 8, grp * 256 + 2048, 16);
      }
    } else {
      for (;;) {
        fence_c();
        u32x2 v;
        if (spins & 7u) v = __builtin_amdgcn_raw_buffer_load_b64(rl, (lane & 31) * 8, grp * 256, HAUX);
        else v = __builtin_amdgcn_raw_buffer_load_b64(rl, (lane & 31) * 8, grp * 256 + 2048, 16);
        if ((lane & 31) == 0 || (lane & 31) >= NG) { v[0] = 0u; v[1] = call; }
        if (__all(v[1] == call)) { mu = max(mu, v[0]); break; }
        if (timed_out(tmo, call, t0, spins)) { mu = 0xffffffffu; break; }
      }
      mu = wave_max(mu);
      __hip_atomic_store((u64*)(peer + WS_GMAX) + (lane >> 3) * 64 + rank * 8 + grp, ((u64)call << 32) | mu,
                         __ATOMIC_RELAXED, __HIP_MEMORY_SCOPE_SYSTEM);
    }
    STAMP(4);
    u64* gm = (u64*)(myws + WS_GMAX) + grp * 64 + lane;
    for (;;) {
      const u64 v = __hip_atomic_load(gm, __ATOMIC_RELAXED, __HIP_MEMORY_SCOPE_SYSTEM);
      if (__all((unsigned)(v >> 32) == call)) { mu = (unsigned)v; break; }
      if (timed_out(tmo, call, t0, spins)) { mu = 0xffffffffu; break; }
    }
    mu = wave_max(mu);
    if (lane == 0) { ctl[18] = mu; ctl[19] = 1u; }
    if (b == 0 && lane == 0)
      __hip_atomic_store((unsigned*)(myws + WS_CTR), call, __ATOMIC_RELAXED, __HIP_MEMORY_SCOPE_AGENT);
    STAMP(5);
  }
  lds_wait(ctl + 19, 1, tmo, call);
  mu = ctl[18];
  const bool bad = mu == 0xffffffffu || ctl[19] != 1u;
  const float amax = __builtin_bit_cast(float, mu);
  const float inv = amax > 0.f ? 127.0f / amax : 0.f;

  const __amdgpu_buffer_rsrc_t ro = mk_rsrc(p.out, NR * MS * NL);
  const int oo = (src * MS + mt * BM + wm * 64 + (lane & 15)) * NL + nt * BN + wn * 64 + (lane >> 4) * 16;
  const int oe = (src * MS + mt * BM + wm * 64 + (lane & 15)) * NL + (NG / 2) * BN + (nt * 2 + wn) * 16 + (lane >> 4) * 4;
#pragma unroll
  for (int i = 0; i < 4; ++i) {
    u32x4 v;
#pragma unroll
    for (int r = 0; r < 4; ++r) {
      unsigned q[4];
#pragma unroll
      for (int j = 0; j < 4; ++j) q[j] = __builtin_bit_cast(unsigned, __builtin_fmaf(acc[i][j][r], inv, 12582912.0f));
      const unsigned d = __builtin_amdgcn_perm(__builtin_amdgcn_perm(q[3], q[2], 0x0c0c0400u),
                                               __builtin_amdgcn_perm(q[1], q[0], 0x0c0c0400u), 0x05040100u);
      v[r] = bad ? 0x80808080u : d;
    }
    __builtin_amdgcn_raw_buffer_store_b128(v, ro, oo, i * 16 * NL, 0);
    unsigned q[4];
#pragma unroll
    for (int r = 0; r < 4; ++r) q[r] = __builtin_bit_cast(unsigned, __builtin_fmaf(acc[i][4][r], inv, 12582912.0f));
    const unsigned d = __builtin_amdgcn_perm(__builtin_amdgcn_perm(q[3], q[2], 0x0c0c0400u),
                                             __builtin_amdgcn_perm(q[1], q[0], 0x0c0c0400u), 0x05040100u);
    if (ex == nt) __builtin_amdgcn_raw_buffer_store_b32(bad ? 0x80808080u : d, ro, oe, i * 16 * NL, 0);
  }
  if (wv == 0) STAMP(6);
}

}

extern "C" void kernel_launch(int rank, int nranks, ncclComm_t comm, hipStream_t stream,
                              void* const* d_in, const int* in_sizes, int n_in, void* d_out,
                              int out_size, const int* in_dtypes, int out_dtype,
                              void* const* d_out_all, void* const* d_ws_all, size_t ws_bytes) {
  (void)comm; (void)in_dtypes; (void)out_dtype; (void)d_out_all;
  if (nranks != NR || (unsigned)rank >= (unsigned)NR || n_in < 2 || ws_bytes < WS_END ||
      in_sizes[0] != MS * KD || in_sizes[1] != KD * NL || out_size != NR * MS * NL) {
    fprintf(stderr, "unexpected problem shape\n");
    return;
  }
  Params p;
  p.x = (const float*)d_in[0];
  p.w = (const float*)d_in[1];
  p.out = (signed char*)d_out;
  for (int i = 0; i < NR; ++i) p.ws[i] = (char*)d_ws_all[(rank + i) % NR];
  p.rank = rank;
  hipLaunchKernelGGL(ag_gemm_q8, dim3(NWG), dim3(NTHREADS), 0, stream, p);
  HIP_CHECK(hipGetLastError());
}
```
